```python
import numpy as np

kernel_name = "gpu_gemm_ar_m8192_k8192_n8192_f32_none"


ABSMAX_RTOL = 0.001

COMPUTE_DTYPE = "float32"
EPILOGUE = "none"
IN_QUANT = None
OUT_QUANT = None
IN_DTYPES = {"x": "float32", "w_mat": "float32"}
WORLD_SIZE = 8

def setup_inputs():

    x = np.random.randn(8192, 8192).astype(np.float32)
    w_mat = (np.random.randn(8192, 8192) / np.sqrt(8192)).astype(np.float32)
    return {"x": x, "w_mat": w_mat}

def _epilogue(y):
    return y

def reference(x, w_mat):
    y = (x @ w_mat).astype(np.float32)
    return _epilogue(y)
```

```cpp
#include <hip/hip_runtime.h>
#include <rccl/rccl.h>
#include <cstdio>

#define HIP_CHECK(expr) do { hipError_t _e = (expr); if (_e != hipSuccess) { \
    fprintf(stderr, "HIP error %s at %s:%d\n", hipGetErrorString(_e), __FILE__, __LINE__); \
    return; } } while (0)

#define LAS __attribute__((address_space(3)))
typedef _Float16 h8 __attribute__((ext_vector_type(8)));
typedef float f32x4 __attribute__((ext_vector_type(4)));
typedef unsigned u32x4 __attribute__((ext_vector_type(4)));

namespace {

constexpr int NR = 8;
constexpr int MM = 8192, NN = 8192, KS = 1024;
constexpr int HTB = 16384;
constexpr size_t BLKB = 524288;
constexpr int NCOMM = 16;
constexpr int NWG = 256;
constexpr int NITEM = 256;
constexpr int AMAX_WG = 512;
constexpr unsigned long long DEADLINE = 100000000ull;

constexpr size_t WS_CTR0 = 0;
constexpr size_t WS_CTR1 = 128;
constexpr size_t WS_TMO = 256;
constexpr size_t WS_AMAXP = 4096;
constexpr size_t WS_XCHG = 8192;
constexpr size_t WS_SPEC = 12288;
constexpr size_t WS_FLAGS = 65536;
constexpr size_t WS_STAGE = 1ull << 20;
constexpr size_t STAGE_PAR = (size_t)NR * 64 * BLKB;
constexpr size_t WS_NEED = WS_STAGE + 2 * STAGE_PAR;

struct Params {
    const float* x;
    const float* w;
    float* out[NR];
    char* ws[NR];
    int rank;
    int phase;
};

__host__ __device__ __forceinline__ int lds_byte(int r, int c) { const int st = (r >> 4) * 2 + (c >> 5), rr = r & 15, cc = c & 31, ob = rr * 64 + cc * 2; return st * 1024 + (ob ^ (((ob >> 9) & 1) << 5)); }
__host__ __device__ __forceinline__ void stage_rc(int b, int& R, int& C) { const int st = b / 1024, sb = b % 1024, swz = sb ^ (((sb >> 9) & 1) << 5); R = (st >> 1) * 16 + swz / 64; C = (st & 1) * 32 + (swz % 64) / 2; }

__device__ __forceinline__ size_t koff(int kt) { const int q = kt >> 5, p = (kt >> 2) & 7, tt = kt & 3; return ((size_t)(p * 64) * 16 + 4 * q + tt) * 32768ull; }

__device__ __forceinline__ void item_of(int n, int& blk, int& q) {
    if (n < 124) { q = n / 31; const int r = n % 31; blk = (r < 15) ? r : (32 + r - 15); }
    else if (n < 188) { const int m = n - 124; q = m >> 4; blk = 48 + (m & 15); }
    else { const int m = n - 188; q = m / 17; blk = 15 + (m % 17); }
}
__device__ __forceinline__ int item_index(int blk, int q) { return blk * 4 + q; }

__device__ __forceinline__ unsigned ld_sys(const unsigned* p) { return __hip_atomic_load(p, __ATOMIC_RELAXED, __HIP_MEMORY_SCOPE_SYSTEM); }
__device__ __forceinline__ unsigned ld_agent(const unsigned* p) { return __hip_atomic_load(p, __ATOMIC_RELAXED, __HIP_MEMORY_SCOPE_AGENT); }

__device__ __forceinline__ int scale_exp(unsigned bits) {
    if (bits == 0u || bits >= 0x7f800000u) return 14;
    int e = (int)(bits >> 23) - 127;
    if ((bits >> 23) == 0u) e = -118 - __builtin_clz(bits);
    return e;
}

__device__ __forceinline__ h8 cvt8(f32x4 a, f32x4 b, int sh) {
    h8 r;
    r[0] = (_Float16)__builtin_amdgcn_ldexpf(a[0], sh); r[1] = (_Float16)__builtin_amdgcn_ldexpf(a[1], sh);
    r[2] = (_Float16)__builtin_amdgcn_ldexpf(a[2], sh); r[3] = (_Float16)__builtin_amdgcn_ldexpf(a[3], sh);
    r[4] = (_Float16)__builtin_amdgcn_ldexpf(b[0], sh); r[5] = (_Float16)__builtin_amdgcn_ldexpf(b[1], sh);
    r[6] = (_Float16)__builtin_amdgcn_ldexpf(b[2], sh); r[7] = (_Float16)__builtin_amdgcn_ldexpf(b[3], sh);
    return r;
}

struct Unit { int pi, pj; int need_x, need_w; int special; };

}

__global__ __launch_bounds__(512, 2) void fused_kernel(const Params P) {
    extern __shared__ __attribute__((aligned(16))) unsigned char shm_[];
    LAS unsigned char* lds = (LAS unsigned char*)shm_;
    const int tid = threadIdx.x;
    const int rank = P.rank;
    char* myws = P.ws[rank];
    unsigned* tmo = (unsigned*)(myws + WS_TMO);

    if (P.phase == 0) {
        LAS unsigned* sred = (LAS unsigned*)lds;
        const int b = blockIdx.x;
        const float* src = (b < AMAX_WG / 2) ? P.x : P.w;
        const int bb = (b < AMAX_WG / 2) ? b : b - AMAX_WG / 2;
        const size_t per = (size_t)MM * KS / (AMAX_WG / 2);
        const u32x4* s4 = (const u32x4*)(src + (size_t)bb * per);
        unsigned m = 0;
#pragma unroll
        for (int i = 0; i < 16; ++i) {
            u32x4 v = __builtin_nontemporal_load(s4 + i * 512 + tid);
            m = max(m, max(max(v[0] & 0x7fffffffu, v[1] & 0x7fffffffu), max(v[2] & 0x7fffffffu, v[3] & 0x7fffffffu)));
        }
#pragma unroll
        for (int o = 32; o > 0; o >>= 1) m = max(m, (unsigned)__shfl_xor((int)m, o));
        if ((tid & 63) == 0) sred[tid >> 6] = m;
        __syncthreads();
        if (tid == 0) {
#pragma unroll
            for (int i = 1; i < 8; ++i) m = max(m, sred[i]);
            __hip_atomic_store((unsigned*)(myws + WS_AMAXP) + b, m, __ATOMIC_RELAXED, __HIP_MEMORY_SCOPE_AGENT);
            asm volatile("s_waitcnt vmcnt(0)" ::: "memory");
            const unsigned ticket = __hip_atomic_fetch_add((unsigned*)(myws + WS_CTR0), 1u, __ATOMIC_RELEASE, __HIP_MEMORY_SCOPE_AGENT);
            sred[8] = ticket;
        }
        __syncthreads();
        const unsigned ticket = sred[8];
        if (ticket % AMAX_WG == AMAX_WG - 1) {
            const unsigned call = ticket / AMAX_WG + 1;
            __builtin_amdgcn_fence(__ATOMIC_ACQUIRE, "agent");
            asm volatile("s_waitcnt vmcnt(0)" ::: "memory");
            unsigned v = ld_agent((const unsigned*)(myws + WS_AMAXP) + tid);
#pragma unroll
            for (int o = 32; o > 0; o >>= 1) v = max(v, (unsigned)__shfl_xor((int)v, o));
            __syncthreads();
            if ((tid & 63) == 0) sred[tid >> 6] = v;
            __syncthreads();
            if (tid < NR) {
                const unsigned ax = max(max(sred[0], sred[1]), max(sred[2], sred[3]));
                const unsigned aw = max(max(sred[4], sred[5]), max(sred[6], sred[7]));
                unsigned long long* slot = (unsigned long long*)(P.ws[tid] + WS_XCHG + (size_t)rank * 128);
                __hip_atomic_store(slot, ((unsigned long long)call << 32) | ax, __ATOMIC_RELAXED, __HIP_MEMORY_SCOPE_SYSTEM);
                __hip_atomic_store(slot + 1, ((unsigned long long)call << 32) | aw, __ATOMIC_RELAXED, __HIP_MEMORY_SCOPE_SYSTEM);
            }
        }
        return;
    }

    LAS unsigned* sctl = (LAS unsigned*)(lds + 8 * HTB);
    const int wid = __builtin_amdgcn_readfirstlane(tid >> 6), lane = tid & 63;
    if (wid == 0) {
        unsigned call = 0;
        if (lane == 0) call = __hip_atomic_fetch_add((unsigned*)(myws + WS_CTR1), 1u, __ATOMIC_RELAXED, __HIP_MEMORY_SCOPE_AGENT) / NWG + 1;
        call = __builtin_amdgcn_readfirstlane(call);
        const unsigned long long* slot = (const unsigned long long*)(myws + WS_XCHG + (size_t)(lane >> 1 & 7) * 128) + (lane & 1);
        unsigned long long v = 0;
        const unsigned long long t0 = __builtin_amdgcn_s_memrealtime();
        for (;;) {
            v = __hip_atomic_load(slot, __ATOMIC_RELAXED, __HIP_MEMORY_SCOPE_SYSTEM);
            if (__all((unsigned)(v >> 32) >= call)) break;
            if (ld_agent(tmo) == call || __builtin_amdgcn_s_memrealtime() - t0 > DEADLINE) {
                if (lane == 0) __hip_atomic_store(tmo, call, __ATOMIC_RELAXED, __HIP_MEMORY_SCOPE_AGENT);
                break;
            }
            __builtin_amdgcn_s_sleep(2);
        }
        unsigned a = (unsigned)v;
        a = max(a, (unsigned)__shfl_xor((int)a, 2)); a = max(a, (unsigned)__shfl_xor((int)a, 4)); a = max(a, (unsigned)__shfl_xor((int)a, 8));
        const unsigned ax = __shfl((int)a, 0), aw = __shfl((int)a, 1);
        if (lane == 0) { sctl[0] = call; sctl[1] = (unsigned)(14 - scale_exp(ax)); sctl[2] = (unsigned)(14 - scale_exp(aw)); }
    }
    __syncthreads();
    const unsigned call = __builtin_amdgcn_readfirstlane(sctl[0]);
    const int shx = (int)__builtin_amdgcn_readfirstlane(sctl[1]), shw = (int)__builtin_amdgcn_readfirstlane(sctl[2]);
    const size_t stage_off = WS_STAGE + (size_t)(call & 1) * STAGE_PAR;

    const int bid = blockIdx.x;
    if (bid < NCOMM) {
        for (int n = bid; n < NITEM; n += NCOMM) {
            int blk, q; item_of(n, blk, q);
            const size_t ioff = stage_off + ((size_t)rank * 64 + blk) * BLKB + (size_t)(4 * q) * 32768;
            if (blk < 32) {
#pragma unroll 1
                for (int kk = 0; kk < 4; ++kk) {
                    h8 v[4];
#pragma unroll
                    for (int c = 0; c < 4; ++c) {
                        int R, C; stage_rc(tid * 16 + (c & 1) * 8192, R, C);
                        const float* s = P.x + (size_t)(blk * 256 + (c >> 1) * 128 + R) * KS + (4 * q + kk) * 64 + C;
                        const f32x4 a = __builtin_nontemporal_load((const f32x4*)s), b2 = __builtin_nontemporal_load((const f32x4*)(s + 4));
                        v[c] = cvt8(a, b2, shx);
                    }
#pragma unroll
                    for (int d = 0; d < NR; ++d) {
                        char* dst = P.ws[(rank + d) & 7] + ioff + (size_t)kk * 32768 + tid * 16;
#pragma unroll
                        for (int c = 0; c < 4; ++c) __builtin_nontemporal_store(v[c], (h8*)(dst + c * 8192));
                    }
                }
            } else {
                const int n4 = lane * 4, kg = wid;
#pragma unroll 1
                for (int kk = 0; kk < 4; ++kk) {
                    const float* s = P.w + (size_t)((4 * q + kk) * 64 + kg * 8) * NN + (blk - 32) * 256 + n4;
                    f32x4 r[8];
#pragma unroll
                    for (int e = 0; e < 8; ++e) r[e] = __builtin_nontemporal_load((const f32x4*)(s + (size_t)e * NN));
                    __syncthreads();
#pragma unroll
                    for (int c = 0; c < 4; ++c) {
                        const f32x4 a = {r[0][c], r[1][c], r[2][c], r[3][c]}, b2 = {r[4][c], r[5][c], r[6][c], r[7][c]};
                        const int nn = n4 + c;
                        *(LAS h8*)(lds + (nn >> 7) * HTB + lds_byte(nn & 127, kg * 8)) = cvt8(a, b2, shw);
                    }
                    __syncthreads();
                    h8 v[4];
#pragma unroll
                    for (int c = 0; c < 4; ++c) v[c] = *(const LAS h8*)(lds + tid * 16 + c * 8192);
#pragma unroll
                    for (int d = 0; d < NR; ++d) {
                        char* dst = P.ws[(rank + d) & 7] + ioff + (size_t)kk * 32768 + tid * 16;
#pragma unroll
                        for (int c = 0; c < 4; ++c) __builtin_nontemporal_store(v[c], (h8*)(dst + c * 8192));
                    }
                }
            }
            asm volatile("s_waitcnt vmcnt(0)" ::: "memory");
            __syncthreads();
            if (wid == 0) {
                __builtin_amdgcn_fence(__ATOMIC_RELEASE, "");
                asm volatile("s_waitcnt vmcnt(0)" ::: "memory");
                if (lane < NR) __hip_atomic_store((unsigned*)(P.ws[lane] + WS_FLAGS + (size_t)item_index(blk, q) * 128) + rank, call, __ATOMIC_RELAXED, __HIP_MEMORY_SCOPE_SYSTEM);
            }
        }
        __syncthreads();
    }

    const int xcd = bid & 7;
    const int idx = (bid < NCOMM) ? 29 : ((bid - NCOMM) >> 3);
    const int prow = idx % 15, pcol = 2 * xcd + idx / 15;
    const bool chosen = (bid >= NCOMM) && (idx == 29);
    const int nunits = (bid < NCOMM) ? ((bid < 8) ? 1 : 0) : 4;
    auto get_unit = [&](int ui) -> Unit {
        Unit u; u.special = 0;
        int k;
        if (bid < NCOMM) { k = 3; u.need_x = 1; u.need_w = 1; }
        else if (chosen) {
            if (ui == 2) { u.pi = 30 + (xcd >> 2); u.pj = 4 * rank + (xcd & 3); u.need_x = 1; u.need_w = 1; u.special = 1; return u; }
            k = (ui == 3) ? 2 : ui; u.need_x = (k != 1); u.need_w = (k != 2);
        } else { k = ui; u.need_x = !(k & 1); u.need_w = (k < 2); }
        u.pi = (k >> 1) * 15 + prow; u.pj = (k & 1) * 16 + pcol;
        return u;
    };

    if (nunits > 0) {
        const int wr = wid >> 2, wc = wid & 3, fr = lane & 15, fq = lane >> 4;
        const unsigned voff0 = tid * 16, voff1 = tid * 16 + 8192;
        const unsigned ldsw = (unsigned)wid * 1024u;
        const int aoff = lds_byte(wr * 64 + fr, fq * 8), boff = lds_byte(wc * 32 + fr, fq * 8);
        const char* stage = myws + stage_off;
        const int osh = -(shx + shw);
#define SA(b, h) (((b) * 2 + (h)) * HTB)
#define SB(b, h) ((4 + (b) * 2 + (h)) * HTB)
#define STAGE(bufoff, gbase) do { \
        __builtin_amdgcn_global_load_lds((const unsigned*)((const char*)(gbase) + voff0), (LAS unsigned*)(lds + (bufoff) + ldsw), 16, 0, 0); \
        __builtin_amdgcn_global_load_lds((const unsigned*)((const char*)(gbase) + voff1), (LAS unsigned*)(lds + (bufoff) + ldsw + 8192), 16, 0, 0); } while (0)
#define LDA(dst, b, h) do { _Pragma("unroll") for (int m = 0; m < 4; ++m) _Pragma("unroll") for (int k = 0; k < 2; ++k) dst[m][k] = *(const LAS h8*)(lds + SA(b, h) + aoff + m * 2048 + k * 1024); } while (0)
#define LDB(dst, b, h) do { _Pragma("unroll") for (int n = 0; n < 2; ++n) _Pragma("unroll") for (int k = 0; k < 2; ++k) dst[n][k] = *(const LAS h8*)(lds + SB(b, h) + boff + n * 2048 + k * 1024); } while (0)
#define MMA(ai, bj, At, Bt) do { __builtin_amdgcn_s_setprio(1); _Pragma("unroll") for (int m = 0; m < 4; ++m) _Pragma("unroll") for (int n = 0; n < 2; ++n) _Pragma("unroll") for (int k = 0; k < 2; ++k) \
        acc[ai][bj][m][n] = __builtin_amdgcn_mfma_f32_16x16x32_f16(Bt[n][k], At[m][k], acc[ai][bj][m][n], 0, 0, 0); __builtin_amdgcn_s_setprio(0); } while (0)
#define WAIT_V(n) asm volatile("s_waitcnt vmcnt(" #n ")" ::: "memory")
#define WAIT_L(n) asm volatile("s_waitcnt lgkmcnt(" #n ")" ::: "memory")
#define BAR __builtin_amdgcn_s_barrier()
#define SCHED __builtin_amdgcn_sched_barrier(0)
        auto ready = [&](const Unit& u, int q) {
            if (!(u.need_x | u.need_w)) return;
            if (wid == 0) {
                const int it = item_index((lane < 8) ? u.pi : 32 + u.pj, q);
                const bool need = (lane < 16) && ((lane < 8) ? u.need_x : u.need_w);
                const unsigned* f = (const unsigned*)(myws + WS_FLAGS + (size_t)it * 128) + (lane & 7);
                const unsigned long long t0 = __builtin_amdgcn_s_memrealtime();
                for (;;) {
                    const unsigned v = need ? ld_sys(f) : call;
                    if (__all(v >= call)) break;
                    if (ld_agent(tmo) == call || __builtin_amdgcn_s_memrealtime() - t0 > DEADLINE) {
                        if (lane == 0) __hip_atomic_store(tmo, call, __ATOMIC_RELAXED, __HIP_MEMORY_SCOPE_AGENT);
                        break;
                    }
                    __builtin_amdgcn_s_sleep(1);
                }
                __builtin_amdgcn_fence(__ATOMIC_ACQUIRE, "");
                asm volatile("s_waitcnt vmcnt(0)" ::: "memory");
            }
            asm volatile("" ::: "memory"); BAR; asm volatile("" ::: "memory");
        };

        f32x4 acc[2][2][4][2];
#pragma unroll
        for (int a = 0; a < 2; ++a)
#pragma unroll
            for (int b = 0; b < 2; ++b)
#pragma unroll
                for (int m = 0; m < 4; ++m)
#pragma unroll
                    for (int n = 0; n < 2; ++n) acc[a][b][m][n] = (f32x4){0.f, 0.f, 0.f, 0.f};
        h8 At[4][2], B0[2][2], B1[2][2];
        Unit cur = get_unit(0);
        const char* cA = stage + (size_t)cur.pi * BLKB; const char* cB = stage + (size_t)(32 + cur.pj) * BLKB;
        ready(cur, 0);
        STAGE(SB(0, 0), cB); STAGE(SA(0, 0), cA); STAGE(SB(0, 1), cB + HTB); STAGE(SA(0, 1), cA + HTB);
        if (wr == 1) BAR;
        WAIT_V(4); BAR;
        STAGE(SB(1, 0), cB + 32768); STAGE(SA(1, 0), cA + 32768); STAGE(SB(1, 1), cB + 32768 + HTB);
        WAIT_V(6); BAR;
        for (int ui = 0;; ++ui) {
            const bool has_next = ui + 1 < nunits;
            const Unit nxt = has_next ? get_unit(ui + 1) : cur;
            const char* nA = stage + (size_t)nxt.pi * BLKB; const char* nB = stage + (size_t)(32 + nxt.pj) * BLKB;
            for (int t = 0; t < 128; t += 2) {
                const bool last = (t == 126);
                const char* a1 = cA + koff(t) + 32768;
                const size_t o2 = last ? 0 : koff(t + 2);
                const char* a2 = (last ? nA : cA) + o2; const char* b2 = (last ? nB : cB) + o2;
                const char* a3 = a2 + 32768; const char* b3 = b2 + 32768;
                if (last) { if (has_next) ready(nxt, 0); }
                else if (((t + 2) & 31) == 0) ready(cur, (t + 2) >> 5);
                LDB(B0, 0, 0); SCHED; LDA(At, 0, 0); STAGE(SA(1, 1), a1 + HTB);
                WAIT_L(8); BAR; WAIT_L(0); MMA(0, 0, At, B0); BAR; SCHED;
                LDB(B1, 0, 1); STAGE(SB(0, 0), b2);
                BAR; WAIT_L(0); MMA(0, 1, At, B1); BAR;
                LDA(At, 0, 1); STAGE(SA(0, 0), a2);
                BAR; WAIT_L(0); MMA(1, 0, At, B0); BAR; SCHED;
                STAGE(SB(0, 1), b2 + HTB);
                WAIT_V(6); BAR; MMA(1, 1, At, B1); BAR;
                LDB(B0, 1, 0); SCHED; LDA(At, 1, 0); STAGE(SA(0, 1), a2 + HTB);
                WAIT_L(8); BAR; WAIT_L(0); MMA(0, 0, At, B0); BAR; SCHED;
                LDB(B1, 1, 1); STAGE(SB(1, 0), b3);
                BAR; WAIT_L(0); MMA(0, 1, At, B1); BAR;
                LDA(At, 1, 1); STAGE(SA(1, 0), a3);
                BAR; WAIT_L(0); MMA(1, 0, At, B0); BAR; SCHED;
                STAGE(SB(1, 1), b3 + HTB);
                WAIT_V(6); BAR; MMA(1, 1, At, B1); BAR;
            }
            {
                const bool bad = (ld_agent(tmo) == call);
                const float qnan = __builtin_nanf("");
#pragma unroll
                for (int ai = 0; ai < 2; ++ai)
#pragma unroll
                    for (int bj = 0; bj < 2; ++bj)
#pragma unroll
                        for (int m = 0; m < 4; ++m)
#pragma unroll
                            for (int n = 0; n < 2; ++n)
#pragma unroll
                                for (int j = 0; j < 4; ++j) acc[ai][bj][m][n][j] = bad ? qnan : __builtin_amdgcn_ldexpf(acc[ai][bj][m][n][j], osh);
                const size_t off0 = (size_t)(cur.pi * 256 + wr * 64 + fr) * NN + cur.pj * 256 + wc * 32 + 4 * fq;
                const int nd = cur.special ? NR : 1;
#pragma unroll 1
                for (int d = 0; d < nd; ++d) {
                    float* C = P.out[(rank + d) & 7] + off0;
#pragma unroll
                    for (int ai = 0; ai < 2; ++ai)
#pragma unroll
                        for (int m = 0; m < 4; ++m) { float* rowp = C + (size_t)(ai * 128 + m * 16) * NN;
#pragma unroll
                            for (int bj = 0; bj < 2; ++bj)
#pragma unroll
                                for (int n = 0; n < 2; ++n) *(f32x4*)(rowp + bj * 128 + n * 16) = acc[ai][bj][m][n]; }
                }
                if (cur.special) {
                    asm volatile("s_waitcnt vmcnt(0)" ::: "memory");
                    __builtin_amdgcn_fence(__ATOMIC_RELEASE, "");
                    asm volatile("s_waitcnt vmcnt(0)" ::: "memory");
                    if (lane < NR && lane != rank)
                        __hip_atomic_fetch_add((unsigned*)(P.ws[lane] + WS_SPEC + (size_t)rank * 128), 1u, __ATOMIC_RELAXED, __HIP_MEMORY_SCOPE_SYSTEM);
                }
            }
            if (!has_next) break;
#pragma unroll
            for (int a = 0; a < 2; ++a)
#pragma unroll
                for (int b = 0; b < 2; ++b)
#pragma unroll
                    for (int m = 0; m < 4; ++m)
#pragma unroll
                        for (int n = 0; n < 2; ++n) acc[a][b][m][n] = (f32x4){0.f, 0.f, 0.f, 0.f};
            cur = nxt; cA = nA; cB = nB;
        }
        WAIT_V(0);
        if (wr == 0) BAR;
        BAR;
    }

    if (bid == NCOMM - 1 && wid == 0) {
        const unsigned need = call * 64u;
        const unsigned* f = (const unsigned*)(myws + WS_SPEC + (size_t)(lane & 7) * 128);
        const bool mine = (lane < NR) && (lane != rank);
        const unsigned long long t0 = __builtin_amdgcn_s_memrealtime();
        bool to = false;
        for (;;) {
            const unsigned v = mine ? ld_sys(f) : need;
            if (__all(v >= need)) break;
            if (ld_agent(tmo) == call || __builtin_amdgcn_s_memrealtime() - t0 > DEADLINE) {
                if (lane == 0) __hip_atomic_store(tmo, call, __ATOMIC_RELAXED, __HIP_MEMORY_SCOPE_AGENT);
                to = true;
                break;
            }
            __builtin_amdgcn_s_sleep(4);
        }
        __builtin_amdgcn_fence(__ATOMIC_ACQUIRE, "");
        if (to && lane == 0) P.out[rank][(size_t)30 * 256 * NN + (size_t)4 * rank * 256] = __builtin_nanf("");
    }
}

extern "C" void kernel_launch(int rank, int nranks, ncclComm_t comm,
                              hipStream_t stream,
                              void* const* d_in, const int* in_sizes, int n_in,
                              void* d_out, int out_size,
                              const int* in_dtypes, int out_dtype,
                              void* const* d_out_all,
                              void* const* d_ws_all, size_t ws_bytes) {
    if (nranks != NR || ws_bytes < WS_NEED || (unsigned)rank >= (unsigned)NR) { fprintf(stderr, "unsupported configuration\n"); return; }
    static bool init = false;
    if (!init) {
        HIP_CHECK(hipFuncSetAttribute((const void*)fused_kernel, hipFuncAttributeMaxDynamicSharedMemorySize, 8 * HTB + 256));
        init = true;
    }
    Params P;
    P.x = (const float*)d_in[0];
    P.w = (const float*)d_in[1];
    for (int p = 0; p < NR; ++p) { P.out[p] = (float*)d_out_all[p]; P.ws[p] = (char*)d_ws_all[p]; }
    P.rank = rank;
    P.phase = 0;
    fused_kernel<<<AMAX_WG, 512, 256, stream>>>(P);
    P.phase = 1;
    fused_kernel<<<NWG, 512, 8 * HTB + 256, stream>>>(P);
    HIP_CHECK(hipGetLastError());
}
```

```cpp
#include <hip/hip_runtime.h>
#include <rccl/rccl.h>
#include <cstdio>

#define HIP_CHECK(expr) do { hipError_t _e = (expr); if (_e != hipSuccess) { \
    fprintf(stderr, "HIP error %s at %s:%d\n", hipGetErrorString(_e), __FILE__, __LINE__); \
    return; } } while (0)

#define LAS __attribute__((address_space(3)))
typedef _Float16 h8 __attribute__((ext_vector_type(8)));
typedef float f32x4 __attribute__((ext_vector_type(4)));
typedef unsigned u32x4 __attribute__((ext_vector_type(4)));

namespace {

constexpr int NR = 8;
constexpr int MM = 8192, NN = 8192, KS = 1024;
constexpr int HTB = 16384;
constexpr size_t BLKB = 524288;
constexpr int NCOMM = 16;
constexpr int NWG = 256;
constexpr int NITEM = 256;
constexpr int SROW = 21;
constexpr int NSPEC = 44;
constexpr int AMAX_WG = 512;
constexpr int CVT_WG = 512;
constexpr unsigned long long DEADLINE = 100000000ull;

constexpr size_t WS_CTR0 = 0;
constexpr size_t WS_CTR1 = 128;
constexpr size_t WS_CTR2 = 256;
constexpr size_t WS_TMO = 384;
constexpr size_t WS_SCL = 512;
constexpr size_t WS_AMAXP = 4096;
constexpr size_t WS_XCHG = 8192;
constexpr size_t WS_SPEC = 12288;
constexpr size_t WS_SDONE = 16384;
constexpr size_t WS_FLAGS = 65536;
constexpr size_t WS_STAGE = 1ull << 20;
constexpr size_t STAGE_PAR = (size_t)NR * 64 * BLKB;
constexpr size_t WS_PROF = WS_STAGE + 2 * STAGE_PAR;
constexpr size_t WS_NEED = WS_PROF + (1ull << 20);

struct Params {
    const float* x;
    const float* w;
    float* out[NR];
    char* ws[NR];
    int rank;
    int phase;
};

__host__ __device__ __forceinline__ int lds_byte(int r, int c) { const int st = (r >> 4) * 2 + (c >> 5), rr = r & 15, cc = c & 31, ob = rr * 64 + cc * 2; return st * 1024 + (ob ^ (((ob >> 9) & 1) << 5)); }
__host__ __device__ __forceinline__ void stage_rc(int b, int& R, int& C) { const int st = b / 1024, sb = b % 1024, swz = sb ^ (((sb >> 9) & 1) << 5); R = (st >> 1) * 16 + swz / 64; C = (st & 1) * 32 + (swz % 64) / 2; }

__device__ __forceinline__ void item_of(int n, int& blk, int& q) {
    if (n < 124) { q = n / 31; const int r = n % 31; blk = (r < 15) ? r : (32 + r - 15); }
    else if (n < 232) { const int m = n - 124; q = m / 27; const int r = m % 27; blk = (r < 16) ? 48 + r : SROW + (r - 16); }
    else { const int m = n - 232; q = m / 6; blk = 15 + (m % 6); }
}
__device__ __forceinline__ unsigned dest_mask(int blk) {
    if (blk < SROW || blk >= 32) return 0xffu;
    const int lo = ((blk - SROW) * 32) / NSPEC, hi = ((blk - SROW) * 32 + 31) / NSPEC;
    return (1u << lo) | (1u << hi);
}
__device__ __forceinline__ int item_index(int blk, int q) { return blk * 4 + q; }

#define STV 1
__device__ __forceinline__ void st_sys(char* p, u32x4 v) {
    asm volatile("global_store_dwordx4 %0, %1, off sc0 sc1\n\ts_nop 1" :: "v"(p), "v"(v) : "memory");
}
__device__ __forceinline__ unsigned ld_sys(const unsigned* p) { return __hip_atomic_load(p, __ATOMIC_RELAXED, __HIP_MEMORY_SCOPE_SYSTEM); }
__device__ __forceinline__ unsigned ld_agent(const unsigned* p) { return __hip_atomic_load(p, __ATOMIC_RELAXED, __HIP_MEMORY_SCOPE_AGENT); }

__device__ __forceinline__ int scale_exp(unsigned bits) {
    if (bits == 0u || bits >= 0x7f800000u) return 14;
    int e = (int)(bits >> 23) - 127;
    if ((bits >> 23) == 0u) e = -118 - __builtin_clz(bits);
    return e;
}

__device__ __forceinline__ h8 cvt8(f32x4 a, f32x4 b, int sh) {
    h8 r;
    r[0] = (_Float16)__builtin_amdgcn_ldexpf(a[0], sh); r[1] = (_Float16)__builtin_amdgcn_ldexpf(a[1], sh);
    r[2] = (_Float16)__builtin_amdgcn_ldexpf(a[2], sh); r[3] = (_Float16)__builtin_amdgcn_ldexpf(a[3], sh);
    r[4] = (_Float16)__builtin_amdgcn_ldexpf(b[0], sh); r[5] = (_Float16)__builtin_amdgcn_ldexpf(b[1], sh);
    r[6] = (_Float16)__builtin_amdgcn_ldexpf(b[2], sh); r[7] = (_Float16)__builtin_amdgcn_ldexpf(b[3], sh);
    return r;
}

enum { K_LOCAL = 0, K_REMOTE = 1, K_FULL = 2 };
struct Unit { int pi, pj; int need_x, need_w; int kind; int special; };

}

__global__ __launch_bounds__(512, 2) void fused_kernel(const Params P) {
    extern __shared__ __attribute__((aligned(16))) unsigned char shm_[];
    LAS unsigned char* lds = (LAS unsigned char*)shm_;
    const int tid = threadIdx.x;
    const int rank = P.rank;
    char* myws = P.ws[rank];
    unsigned* tmo = (unsigned*)(myws + WS_TMO);
    const int wid = __builtin_amdgcn_readfirstlane(tid >> 6), lane = tid & 63;
    const int bid = blockIdx.x;

    if (P.phase == 0) {
        LAS unsigned* sred = (LAS unsigned*)lds;
        const float* src = (bid < AMAX_WG / 2) ? P.x : P.w;
        const int bb = (bid < AMAX_WG / 2) ? bid : bid - AMAX_WG / 2;
        const size_t per = (size_t)MM * KS / (AMAX_WG / 2);
        const u32x4* s4 = (const u32x4*)(src + (size_t)bb * per);
        unsigned m = 0;
#pragma unroll
        for (int i = 0; i < 16; ++i) {
            u32x4 v = s4[i * 512 + tid];
            m = max(m, max(max(v[0] & 0x7fffffffu, v[1] & 0x7fffffffu), max(v[2] & 0x7fffffffu, v[3] & 0x7fffffffu)));
        }
#pragma unroll
        for (int o = 32; o > 0; o >>= 1) m = max(m, (unsigned)__shfl_xor((int)m, o));
        if (lane == 0) sred[wid] = m;
        __syncthreads();
        if (tid == 0) {
#pragma unroll
            for (int i = 1; i < 8; ++i) m = max(m, sred[i]);
            __hip_atomic_store((unsigned*)(myws + WS_AMAXP) + bid, m, __ATOMIC_RELAXED, __HIP_MEMORY_SCOPE_AGENT);
            asm volatile("s_waitcnt vmcnt(0)" ::: "memory");
            sred[8] = __hip_atomic_fetch_add((unsigned*)(myws + WS_CTR0), 1u, __ATOMIC_RELEASE, __HIP_MEMORY_SCOPE_AGENT);
        }
        __syncthreads();
        const unsigned ticket = sred[8];
        if (ticket % AMAX_WG == AMAX_WG - 1) {
            const unsigned call = ticket / AMAX_WG + 1;
            __builtin_amdgcn_fence(__ATOMIC_ACQUIRE, "agent");
            asm volatile("s_waitcnt vmcnt(0)" ::: "memory");
            unsigned v = ld_agent((const unsigned*)(myws + WS_AMAXP) + tid);
#pragma unroll
            for (int o = 32; o > 0; o >>= 1) v = max(v, (unsigned)__shfl_xor((int)v, o));
            __syncthreads();
            if (lane == 0) sred[wid] = v;
            __syncthreads();
            if (tid < NR) {
                const unsigned ax = max(max(sred[0], sred[1]), max(sred[2], sred[3]));
                const unsigned aw = max(max(sred[4], sred[5]), max(sred[6], sred[7]));
                unsigned long long* slot = (unsigned long long*)(P.ws[tid] + WS_XCHG + (size_t)rank * 128);
                __hip_atomic_store(slot, ((unsigned long long)call << 32) | ax, __ATOMIC_RELAXED, __HIP_MEMORY_SCOPE_SYSTEM);
                __hip_atomic_store(slot + 1, ((unsigned long long)call << 32) | aw, __ATOMIC_RELAXED, __HIP_MEMORY_SCOPE_SYSTEM);
            }
        }
        return;
    }

    if (P.phase == 1) {
        LAS unsigned* sctl = (LAS unsigned*)(lds + 2 * HTB);
        if (wid == 0) {
            unsigned call = 0;
            if (lane == 0) call = __hip_atomic_fetch_add((unsigned*)(myws + WS_CTR1), 1u, __ATOMIC_RELAXED, __HIP_MEMORY_SCOPE_AGENT) / CVT_WG + 1;
            call = __builtin_amdgcn_readfirstlane(call);
            const unsigned long long* slot = (const unsigned long long*)(myws + WS_XCHG + (size_t)(lane >> 1 & 7) * 128) + (lane & 1);
            unsigned long long v = 0;
            const unsigned long long t0 = __builtin_amdgcn_s_memrealtime();
            for (;;) {
                v = __hip_atomic_load(slot, __ATOMIC_RELAXED, __HIP_MEMORY_SCOPE_SYSTEM);
                if (__all((unsigned)(v >> 32) >= call)) break;
                if (ld_agent(tmo) == call || __builtin_amdgcn_s_memrealtime() - t0 > DEADLINE) {
                    if (lane == 0) __hip_atomic_store(tmo, call, __ATOMIC_RELAXED, __HIP_MEMORY_SCOPE_AGENT);
                    break;
                }
                __builtin_amdgcn_s_sleep(2);
            }
            unsigned a = (unsigned)v;
            a = max(a, (unsigned)__shfl_xor((int)a, 2)); a = max(a, (unsigned)__shfl_xor((int)a, 4)); a = max(a, (unsigned)__shfl_xor((int)a, 8));
            const unsigned ax = __shfl((int)a, 0), aw = __shfl((int)a, 1);
            if (lane == 0) {
                sctl[0] = call; sctl[1] = (unsigned)(14 - scale_exp(ax)); sctl[2] = (unsigned)(14 - scale_exp(aw));
                if (bid == 0) { ((int*)(myws + WS_SCL))[0] = 14 - scale_exp(ax); ((int*)(myws + WS_SCL))[1] = 14 - scale_exp(aw); }
            }
        }
        __syncthreads();
        const unsigned call = __builtin_amdgcn_readfirstlane(sctl[0]);
        const int shx = (int)__builtin_amdgcn_readfirstlane(sctl[1]), shw = (int)__builtin_amdgcn_readfirstlane(sctl[2]);
        char* img = myws + WS_STAGE + (size_t)(call & 1) * STAGE_PAR + (size_t)rank * 64 * BLKB;
        for (int u = bid; u < 1024; u += CVT_WG) {
            const int blk = u >> 4, kk = u & 15;
            char* dst = img + (size_t)u * 32768 + tid * 16;
            if (blk < 32) {
#pragma unroll
                for (int c = 0; c < 4; ++c) {
                    int R, C; stage_rc(tid * 16 + (c & 1) * 8192, R, C);
                    const float* s = P.x + (size_t)(blk * 256 + (c >> 1) * 128 + R) * KS + kk * 64 + C;
                    *(h8*)(dst + c * 8192) = cvt8(*(const f32x4*)s, *(const f32x4*)(s + 4), shx);
                }
            } else {
                const int n4 = lane * 4;
                const float* s = P.w + (size_t)(kk * 64 + wid * 8) * NN + (blk - 32) * 256 + n4;
                f32x4 r[8];
#pragma unroll
                for (int e = 0; e < 8; ++e) r[e] = *(const f32x4*)(s + (size_t)e * NN);
                __syncthreads();
#pragma unroll
                for (int c = 0; c < 4; ++c) {
                    const f32x4 a = {r[0][c], r[1][c], r[2][c], r[3][c]}, b2 = {r[4][c], r[5][c], r[6][c], r[7][c]};
                    const int nn = n4 + c;
                    *(LAS h8*)(lds + (nn >> 7) * HTB + lds_byte(nn & 127, wid * 8)) = cvt8(a, b2, shw);
                }
                __syncthreads();
#pragma unroll
                for (int c = 0; c < 4; ++c) *(h8*)(dst + c * 8192) = *(const LAS h8*)(lds + tid * 16 + c * 8192);
            }
        }
        return;
    }

    LAS unsigned* sctl = (LAS unsigned*)(lds + 8 * HTB);
#define STAMP() do {} while (0)
    STAMP();
    if (tid == 0) sctl[0] = __hip_atomic_fetch_add((unsigned*)(myws + WS_CTR2), 1u, __ATOMIC_RELAXED, __HIP_MEMORY_SCOPE_AGENT) / NWG + 1;
    __syncthreads();
    const unsigned call = __builtin_amdgcn_readfirstlane(sctl[0]);
    const int osh = -(__builtin_amdgcn_readfirstlane(((const int*)(myws + WS_SCL))[0]) + __builtin_amdgcn_readfirstlane(((const int*)(myws + WS_SCL))[1]));
    const size_t stage_off = WS_STAGE + (size_t)(call & 1) * STAGE_PAR;

    const int xcd = bid & 7, slot = bid >> 3;
    const bool is_comm = bid < NCOMM;
    if (is_comm) {
        for (int n = bid; n < NITEM; n += NCOMM) {
            int blk, q; item_of(n, blk, q);
            const unsigned dm = dest_mask(blk) & ~(1u << rank);
            if (dm == 0u) continue;
            const size_t ioff = stage_off + ((size_t)rank * 64 + blk) * BLKB + (size_t)(4 * q) * 32768 + tid * 16;
            u32x4 v0 = __builtin_nontemporal_load((const u32x4*)(myws + ioff)), v1 = __builtin_nontemporal_load((const u32x4*)(myws + ioff + 8192));
#pragma unroll 1
            for (int c = 0; c < 8; ++c) {
                const u32x4 w0 = v0, w1 = v1;
                if (c < 7) { v0 = __builtin_nontemporal_load((const u32x4*)(myws + ioff + (c + 1) * 16384)); v1 = __builtin_nontemporal_load((const u32x4*)(myws + ioff + (c + 1) * 16384 + 8192)); }
#pragma unroll
                for (int d = 1; d < NR; ++d) {
                    const int p = (rank + d) & 7;
                    if ((dm >> p) & 1u) {
                        char* dst = P.ws[p] + ioff + c * 16384;
                        st_sys(dst, w0); st_sys(dst + 8192, w1);
                    }
                }
            }
            asm volatile("s_waitcnt vmcnt(0)" ::: "memory");
            __syncthreads();
            if (wid == 0 && lane < NR && ((dm >> lane) & 1u))
                __hip_atomic_store((unsigned*)(P.ws[lane] + WS_FLAGS + (size_t)item_index(blk, q) * 128) + rank, call, __ATOMIC_RELAXED, __HIP_MEMORY_SCOPE_SYSTEM);
            STAMP();
        }
        for (int z = bid; z < 4 * NSPEC; z += NCOMM) {
            const int pos = z >> 2, sq = z & 3;
            if (wid == 0) {
                const unsigned* f = (const unsigned*)(myws + WS_SDONE + (size_t)pos * 128);
                const unsigned long long t0 = __builtin_amdgcn_s_memrealtime();
                while (ld_agent(f) < call * 8u) {
                    if (ld_agent(tmo) == call || __builtin_amdgcn_s_memrealtime() - t0 > DEADLINE) {
                        if (lane == 0) __hip_atomic_store(tmo, call, __ATOMIC_RELAXED, __HIP_MEMORY_SCOPE_AGENT);
                        break;
                    }
                    __builtin_amdgcn_s_sleep(2);
                }
                __builtin_amdgcn_fence(__ATOMIC_ACQUIRE, "agent");
                asm volatile("s_waitcnt vmcnt(0)" ::: "memory");
            }
            __syncthreads();
            const int st = NSPEC * rank + pos;
            const size_t ooff = ((size_t)((SROW + (st >> 5)) * 256 + sq * 64 + wid * 8) * NN + (st & 31) * 256 + lane * 4) * 4;
            const bool bad = (ld_agent(tmo) == call);
#pragma unroll 1
            for (int r = 0; r < 8; r += 2) {
                u32x4 w0 = *(const u32x4*)((const char*)P.out[rank] + ooff + (size_t)r * NN * 4), w1 = *(const u32x4*)((const char*)P.out[rank] + ooff + (size_t)(r + 1) * NN * 4);
                if (bad) { w0 = (u32x4){0x7fc00000u, 0x7fc00000u, 0x7fc00000u, 0x7fc00000u}; w1 = w0; }
#pragma unroll
                for (int d = 1; d < NR; ++d) {
                    char* dst = (char*)P.out[(rank + d) & 7] + ooff + (size_t)r * NN * 4;
                    st_sys(dst, w0); st_sys(dst + (size_t)NN * 4, w1);
                }
            }
            asm volatile("s_waitcnt vmcnt(0)" ::: "memory");
            __syncthreads();
            if (wid == 0 && lane < NR && lane != rank)
                __hip_atomic_fetch_add((unsigned*)(P.ws[lane] + WS_SPEC + (size_t)rank * 128), 1u, __ATOMIC_RELAXED, __HIP_MEMORY_SCOPE_SYSTEM);
            STAMP();
        }
    }

    const int pos = xcd * 30 + (slot - NCOMM / 8);
    auto tile_of = [&](int r, int& pi, int& pj, int& special) -> bool {
        special = 0;
        if (r == 0) { pi = (pos % 60) >> 2; pj = (pos / 60) * 4 + (pos & 3); return true; }
        int b;
        if (r == 1) {
            if (pos < NSPEC) { const int st = NSPEC * rank + pos; pi = SROW + (st >> 5); pj = st & 31; special = 1; return true; }
            b = pos - NSPEC;
        } else {
            if (pos >= NSPEC) { const int c = pos - NSPEC; pi = 15 + c % 6; pj = c / 6; return c < 192; }
            b = 240 - NSPEC + pos;
        }
        pi = (b % 60) >> 2; pj = 16 + (b / 60) * 4 + (b & 3);
        return true;
    };
    int pi0, pj0, pi1, pj1, pi2, pj2, sp0, sp1, sp2;
    tile_of(0, pi0, pj0, sp0); tile_of(1, pi1, pj1, sp1);
    const bool has2 = tile_of(2, pi2, pj2, sp2);
    const int nunits = is_comm ? 0 : (has2 ? 5 : 3);
    auto get_unit = [&](int ui) -> Unit {
        Unit u;
        if (!has2 && ui > 0) ui += 1;
        u.special = 0; u.need_x = 1; u.need_w = 1;
        if (ui == 0) { u.pi = pi1; u.pj = pj1; u.kind = K_LOCAL; }
        else if (ui == 1) { u.pi = pi2; u.pj = pj2; u.kind = K_LOCAL; }
        else if (ui == 2) { u.pi = pi0; u.pj = pj0; u.kind = K_FULL; }
        else if (ui == 3) { u.pi = pi1; u.pj = pj1; u.kind = K_REMOTE; u.special = sp1; u.need_x = (pi1 != pi0); u.need_w = (pj1 != pj0); }
        else { u.pi = pi2; u.pj = pj2; u.kind = K_REMOTE; u.need_x = (pi2 != pi0) && (pi2 != pi1); u.need_w = (pj2 != pj0) && (pj2 != pj1); }
        return u;
    };
    auto koff = [&](int kind, int kt) -> size_t {
        if (kind == K_FULL) { if (kt < 16) kind = K_LOCAL; else { kt -= 16; kind = K_REMOTE; } }
        if (kind == K_LOCAL) return ((size_t)(rank * 64) * 16 + kt) * 32768ull;
        const int q = kt / 28, r = kt - q * 28, p = (rank + 1 + (r >> 2)) & 7;
        return ((size_t)(p * 64) * 16 + 4 * q + (r & 3)) * 32768ull;
    };

    if (nunits > 0) {
        const int wr = wid >> 2, wc = wid & 3, fr = lane & 15, fq = lane >> 4;
        const unsigned voff0 = tid * 16, voff1 = tid * 16 + 8192;
        const unsigned ldsw = (unsigned)wid * 1024u;
        const int aoff = lds_byte(wr * 64 + fr, fq * 8), boff = lds_byte(wc * 32 + fr, fq * 8);
        const char* stage = myws + stage_off;
#define SA(b, h) (((b) * 2 + (h)) * HTB)
#define SB(b, h) ((4 + (b) * 2 + (h)) * HTB)
#define STAGE(bufoff, gbase) do { \
        __builtin_amdgcn_global_load_lds((const unsigned*)((const char*)(gbase) + voff0), (LAS unsigned*)(lds + (bufoff) + ldsw), 16, 0, 0); \
        __builtin_amdgcn_global_load_lds((const unsigned*)((const char*)(gbase) + voff1), (LAS unsigned*)(lds + (bufoff) + ldsw + 8192), 16, 0, 0); } while (0)
#define LDA(dst, b, h) do { _Pragma("unroll") for (int m = 0; m < 4; ++m) _Pragma("unroll") for (int k = 0; k < 2; ++k) dst[m][k] = *(const LAS h8*)(lds + SA(b, h) + aoff + m * 2048 + k * 1024); } while (0)
#define LDB(dst, b, h) do { _Pragma("unroll") for (int n = 0; n < 2; ++n) _Pragma("unroll") for (int k = 0; k < 2; ++k) dst[n][k] = *(const LAS h8*)(lds + SB(b, h) + boff + n * 2048 + k * 1024); } while (0)
#define MMA(ai, bj, At, Bt) do { __builtin_amdgcn_s_setprio(1); _Pragma("unroll") for (int m = 0; m < 4; ++m) _Pragma("unroll") for (int n = 0; n < 2; ++n) _Pragma("unroll") for (int k = 0; k < 2; ++k) \
        acc[ai][bj][m][n] = __builtin_amdgcn_mfma_f32_16x16x32_f16(Bt[n][k], At[m][k], acc[ai][bj][m][n], 0, 0, 0); __builtin_amdgcn_s_setprio(0); } while (0)
#define WAIT_V(n) asm volatile("s_waitcnt vmcnt(" #n ")" ::: "memory")
#define WAIT_L(n) asm volatile("s_waitcnt lgkmcnt(" #n ")" ::: "memory")
#define BAR __builtin_amdgcn_s_barrier()
#define SCHED __builtin_amdgcn_sched_barrier(0)
        auto ready = [&](const Unit& u, int q) {
            if (!(u.need_x | u.need_w)) return;
            if (wid == 0) {
                const int it = item_index((lane < 8) ? u.pi : 32 + u.pj, q);
                const bool need = (lane < 16) && ((lane & 7) != rank) && ((lane < 8) ? u.need_x : u.need_w);
                const unsigned* f = (const unsigned*)(myws + WS_FLAGS + (size_t)it * 128) + (lane & 7);
                const unsigned long long t0 = __builtin_amdgcn_s_memrealtime();
                for (;;) {
                    const unsigned v = need ? ld_sys(f) : call;
                    if (__all(v >= call)) break;
                    if (ld_agent(tmo) == call || __builtin_amdgcn_s_memrealtime() - t0 > DEADLINE) {
                        if (lane == 0) __hip_atomic_store(tmo, call, __ATOMIC_RELAXED, __HIP_MEMORY_SCOPE_AGENT);
                        break;
                    }
                    __builtin_amdgcn_s_sleep(1);
                }
                __builtin_amdgcn_fence(__ATOMIC_ACQUIRE, "");
                asm volatile("s_waitcnt vmcnt(0)" ::: "memory");
            }
            asm volatile("" ::: "memory"); BAR; asm volatile("" ::: "memory");
            STAMP();
        };

        f32x4 acc[2][2][4][2];
#pragma unroll
        for (int a = 0; a < 2; ++a)
#pragma unroll
            for (int b = 0; b < 2; ++b)
#pragma unroll
                for (int m = 0; m < 4; ++m)
#pragma unroll
                    for (int n = 0; n < 2; ++n) acc[a][b][m][n] = (f32x4){0.f, 0.f, 0.f, 0.f};
        h8 At[4][2], B0[2][2], B1[2][2];
        Unit cur = get_unit(0);
        const char* cA = stage + (size_t)cur.pi * BLKB; const char* cB = stage + (size_t)(32 + cur.pj) * BLKB;
        {
            const size_t o0 = koff(cur.kind, 0);
            STAGE(SB(0, 0), cB + o0); STAGE(SA(0, 0), cA + o0); STAGE(SB(0, 1), cB + o0 + HTB); STAGE(SA(0, 1), cA + o0 + HTB);
            if (wr == 1) BAR;
            WAIT_V(4); BAR;
            STAGE(SB(1, 0), cB + o0 + 32768); STAGE(SA(1, 0), cA + o0 + 32768); STAGE(SB(1, 1), cB + o0 + 32768 + HTB);
            WAIT_V(6); BAR;
        }
        for (int ui = 0;; ++ui) {
            const bool has_next = ui + 1 < nunits;
            const Unit nxt = has_next ? get_unit(ui + 1) : cur;
            const char* nA = stage + (size_t)nxt.pi * BLKB; const char* nB = stage + (size_t)(32 + nxt.pj) * BLKB;
            const int nt = (cur.kind == K_LOCAL) ? 16 : (cur.kind == K_REMOTE) ? 112 : 128;
            const int rbase = (cur.kind == K_FULL) ? 16 : 0;
            for (int t = 0; t < nt; t += 2) {
                const bool last = (t == nt - 2);
                const char* a1 = cA + koff(cur.kind, t) + 32768;
                const size_t o2 = last ? koff(nxt.kind, 0) : koff(cur.kind, t + 2);
                const char* a2 = (last ? nA : cA) + o2; const char* b2 = (last ? nB : cB) + o2;
                const char* a3 = a2 + 32768; const char* b3 = b2 + 32768;
                if (last) { if (has_next && nxt.kind == K_REMOTE) ready(nxt, 0); }
                else if (cur.kind != K_LOCAL && t + 2 >= rbase && (t + 2 - rbase) % 28 == 0) ready(cur, (t + 2 - rbase) / 28);
                LDB(B0, 0, 0); SCHED; LDA(At, 0, 0); STAGE(SA(1, 1), a1 + HTB);
                WAIT_L(8); BAR; WAIT_L(0); MMA(0, 0, At, B0); BAR; SCHED;
                LDB(B1, 0, 1); STAGE(SB(0, 0), b2);
                BAR; WAIT_L(0); MMA(0, 1, At, B1); BAR;
                LDA(At, 0, 1); STAGE(SA(0, 0), a2);
                BAR; WAIT_L(0); MMA(1, 0, At, B0); BAR; SCHED;
                STAGE(SB(0, 1), b2 + HTB);
                WAIT_V(6); BAR; MMA(1, 1, At, B1); BAR;
                LDB(B0, 1, 0); SCHED; LDA(At, 1, 0); STAGE(SA(0, 1), a2 + HTB);
                WAIT_L(8); BAR; WAIT_L(0); MMA(0, 0, At, B0); BAR; SCHED;
                LDB(B1, 1, 1); STAGE(SB(1, 0), b3);
                BAR; WAIT_L(0); MMA(0, 1, At, B1); BAR;
                LDA(At, 1, 1); STAGE(SA(1, 0), a3);
                BAR; WAIT_L(0); MMA(1, 0, At, B0); BAR; SCHED;
                STAGE(SB(1, 1), b3 + HTB);
                WAIT_V(6); BAR; MMA(1, 1, At, B1); BAR;
            }
            {
                const size_t off0 = (size_t)(cur.pi * 256 + wr * 64 + fr) * NN + cur.pj * 256 + wc * 32 + 4 * fq;
                const bool bad = (cur.kind != K_LOCAL) && (ld_agent(tmo) == call);
                const float qnan = __builtin_nanf("");
                const int sh = (cur.kind == K_LOCAL) ? 0 : osh;
                if (cur.kind == K_REMOTE) {
                    const float* C = P.out[rank] + off0;
#pragma unroll
                    for (int ai = 0; ai < 2; ++ai)
#pragma unroll
                        for (int m = 0; m < 4; ++m) { const float* rowp = C + (size_t)(ai * 128 + m * 16) * NN;
#pragma unroll
                            for (int bj = 0; bj < 2; ++bj)
#pragma unroll
                                for (int n = 0; n < 2; ++n) acc[ai][bj][m][n] += *(const f32x4*)(rowp + bj * 128 + n * 16);
                            asm volatile("" : "+v"(acc[ai][0][m][0]), "+v"(acc[ai][0][m][1]), "+v"(acc[ai][1][m][0]), "+v"(acc[ai][1][m][1]) :: "memory"); }
                }
#pragma unroll
                for (int ai = 0; ai < 2; ++ai)
#pragma unroll
                    for (int bj = 0; bj < 2; ++bj)
#pragma unroll
                        for (int m = 0; m < 4; ++m)
#pragma unroll
                            for (int n = 0; n < 2; ++n)
#pragma unroll
                                for (int j = 0; j < 4; ++j) acc[ai][bj][m][n][j] = bad ? qnan : __builtin_amdgcn_ldexpf(acc[ai][bj][m][n][j], sh);
                float* C = P.out[rank] + off0;
                if (cur.special) {
#pragma unroll
                    for (int ai = 0; ai < 2; ++ai)
#pragma unroll
                        for (int m = 0; m < 4; ++m) { float* rowp = C + (size_t)(ai * 128 + m * 16) * NN;
#pragma unroll
                            for (int bj = 0; bj < 2; ++bj)
#pragma unroll
                                for (int n = 0; n < 2; ++n) st_sys((char*)(rowp + bj * 128 + n * 16), __builtin_bit_cast(u32x4, acc[ai][bj][m][n])); }
                    asm volatile("s_waitcnt vmcnt(0)" ::: "memory");
                    if (lane == 0) __hip_atomic_fetch_add((unsigned*)(myws + WS_SDONE + (size_t)pos * 128), 1u, __ATOMIC_RELAXED, __HIP_MEMORY_SCOPE_AGENT);
                } else {
#pragma unroll
                    for (int ai = 0; ai < 2; ++ai)
#pragma unroll
                        for (int m = 0; m < 4; ++m) { float* rowp = C + (size_t)(ai * 128 + m * 16) * NN;
#pragma unroll
                            for (int bj = 0; bj < 2; ++bj)
#pragma unroll
                                for (int n = 0; n < 2; ++n) *(f32x4*)(rowp + bj * 128 + n * 16) = acc[ai][bj][m][n]; }
                }
            }
            STAMP();
            if (!has_next) break;
#pragma unroll
            for (int a = 0; a < 2; ++a)
#pragma unroll
                for (int b = 0; b < 2; ++b)
#pragma unroll
                    for (int m = 0; m < 4; ++m)
#pragma unroll
                        for (int n = 0; n < 2; ++n) acc[a][b][m][n] = (f32x4){0.f, 0.f, 0.f, 0.f};
            cur = nxt; cA = nA; cB = nB;
        }
        WAIT_V(0);
        if (wr == 0) BAR;
        BAR;
    }

    if (bid == NCOMM - 1 && wid == 0) {
        const unsigned need = call * (unsigned)(4 * NSPEC);
        const unsigned* f = (const unsigned*)(myws + WS_SPEC + (size_t)(lane & 7) * 128);
        const bool mine = (lane < NR) && (lane != rank);
        const unsigned long long t0 = __builtin_amdgcn_s_memrealtime();
        bool to = false;
        for (;;) {
            const unsigned v = mine ? ld_sys(f) : need;
            if (__all(v >= need)) break;
            if (ld_agent(tmo) == call || __builtin_amdgcn_s_memrealtime() - t0 > DEADLINE) {
                if (lane == 0) __hip_atomic_store(tmo, call, __ATOMIC_RELAXED, __HIP_MEMORY_SCOPE_AGENT);
                to = true;
                break;
            }
            __builtin_amdgcn_s_sleep(4);
        }
        __builtin_amdgcn_fence(__ATOMIC_ACQUIRE, "");
        if (to && lane == 0) P.out[rank][0] = __builtin_nanf("");
    }
    STAMP();
}

extern "C" void kernel_launch(int rank, int nranks, ncclComm_t comm,
                              hipStream_t stream,
                              void* const* d_in, const int* in_sizes, int n_in,
                              void* d_out, int out_size,
                              const int* in_dtypes, int out_dtype,
                              void* const* d_out_all,
                              void* const* d_ws_all, size_t ws_bytes) {
    if (nranks != NR || ws_bytes < WS_NEED || (unsigned)rank >= (unsigned)NR) { fprintf(stderr, "unsupported configuration\n"); return; }
    static bool init = false;
    if (!init) {
        HIP_CHECK(hipFuncSetAttribute((const void*)fused_kernel, hipFuncAttributeMaxDynamicSharedMemorySize, 8 * HTB + 256));
        init = true;
    }
    Params P;
    P.x = (const float*)d_in[0];
    P.w = (const float*)d_in[1];
    for (int p = 0; p < NR; ++p) { P.out[p] = (float*)d_out_all[p]; P.ws[p] = (char*)d_ws_all[p]; }
    P.rank = rank;
    P.phase = 0;
    fused_kernel<<<AMAX_WG, 512, 256, stream>>>(P);
    P.phase = 1;
    fused_kernel<<<CVT_WG, 512, 2 * HTB + 256, stream>>>(P);
    P.phase = 2;
    fused_kernel<<<NWG, 512, 8 * HTB + 256, stream>>>(P);
    HIP_CHECK(hipGetLastError());
}
```

```cpp
#include <hip/hip_runtime.h>
#include <rccl/rccl.h>
#include <cstdio>

#define HIP_CHECK(expr) do { hipError_t _e = (expr); if (_e != hipSuccess) { \
    fprintf(stderr, "HIP error %s at %s:%d\n", hipGetErrorString(_e), __FILE__, __LINE__); \
    return; } } while (0)

#define LAS __attribute__((address_space(3)))
typedef _Float16 h8 __attribute__((ext_vector_type(8)));
typedef float f32x4 __attribute__((ext_vector_type(4)));
typedef unsigned u32x4 __attribute__((ext_vector_type(4)));

namespace {

constexpr int NR = 8;
constexpr int MM = 8192, NN = 8192, KS = 1024;
constexpr int HTB = 16384;
constexpr size_t BLKB = 524288;
constexpr int NCOMM = 16;
constexpr int NWG = 256;
constexpr int NITEM = 256;
constexpr int SROW = 21;
constexpr int NSPEC = 44;
constexpr int PRE_WG = 256;
constexpr unsigned long long DEADLINE = 100000000ull;

constexpr size_t WS_CTR0 = 0;
constexpr size_t WS_CTR2 = 256;
constexpr size_t WS_TMO = 384;
constexpr size_t WS_SCL = 512;
constexpr size_t WS_AMAXP = 4096;
constexpr size_t WS_XCHG = 8192;
constexpr size_t WS_SPEC = 12288;
constexpr size_t WS_SDONE = 16384;
constexpr size_t WS_FLAGS = 65536;
constexpr size_t WS_STAGE = 1ull << 20;
constexpr size_t STAGE_PAR = (size_t)NR * 64 * BLKB;
constexpr size_t WS_PROF = WS_STAGE + 2 * STAGE_PAR;
constexpr size_t WS_NEED = WS_PROF + (1ull << 20);

struct Params {
    const float* x;
    const float* w;
    float* out[NR];
    char* ws[NR];
    int rank;
    int phase;
};

__host__ __device__ __forceinline__ int lds_byte(int r, int c) { const int st = (r >> 4) * 2 + (c >> 5), rr = r & 15, cc = c & 31, ob = rr * 64 + cc * 2; return st * 1024 + (ob ^ (((ob >> 9) & 1) << 5)); }
__host__ __device__ __forceinline__ void stage_rc(int b, int& R, int& C) { const int st = b / 1024, sb = b % 1024, swz = sb ^ (((sb >> 9) & 1) << 5); R = (st >> 1) * 16 + swz / 64; C = (st & 1) * 32 + (swz % 64) / 2; }

__device__ __forceinline__ void item_of(int n, int& blk, int& q) {
    if (n < 124) { q = n / 31; const int r = n % 31; blk = (r < 15) ? r : (32 + r - 15); }
    else if (n < 232) { const int m = n - 124; q = m / 27; const int r = m % 27; blk = (r < 16) ? 48 + r : SROW + (r - 16); }
    else { const int m = n - 232; q = m / 6; blk = 15 + (m % 6); }
}
__device__ __forceinline__ unsigned dest_mask(int blk) {
    if (blk < SROW || blk >= 32) return 0xffu;
    const int lo = ((blk - SROW) * 32) / NSPEC, hi = ((blk - SROW) * 32 + 31) / NSPEC;
    return (1u << lo) | (1u << hi);
}
__device__ __forceinline__ int item_index(int blk, int q) { return blk * 4 + q; }

__device__ __forceinline__ void st_sys(char* p, u32x4 v) { asm volatile("global_store_dwordx4 %0, %1, off sc0 sc1\n\ts_nop 1" :: "v"(p), "v"(v) : "memory"); }
__device__ __forceinline__ unsigned ld_sys(const unsigned* p) { return __hip_atomic_load(p, __ATOMIC_RELAXED, __HIP_MEMORY_SCOPE_SYSTEM); }
__device__ __forceinline__ unsigned ld_agent(const unsigned* p) { return __hip_atomic_load(p, __ATOMIC_RELAXED, __HIP_MEMORY_SCOPE_AGENT); }

__device__ __forceinline__ int scale_exp(unsigned bits) {
    if (bits == 0u || bits >= 0x7f800000u) return 14;
    int e = (int)(bits >> 23) - 127;
    if ((bits >> 23) == 0u) e = -118 - __builtin_clz(bits);
    return e;
}

__device__ __forceinline__ h8 cvt8(f32x4 a, f32x4 b, int sh) {
    h8 r;
    r[0] = (_Float16)__builtin_amdgcn_ldexpf(a[0], sh); r[1] = (_Float16)__builtin_amdgcn_ldexpf(a[1], sh);
    r[2] = (_Float16)__builtin_amdgcn_ldexpf(a[2], sh); r[3] = (_Float16)__builtin_amdgcn_ldexpf(a[3], sh);
    r[4] = (_Float16)__builtin_amdgcn_ldexpf(b[0], sh); r[5] = (_Float16)__builtin_amdgcn_ldexpf(b[1], sh);
    r[6] = (_Float16)__builtin_amdgcn_ldexpf(b[2], sh); r[7] = (_Float16)__builtin_amdgcn_ldexpf(b[3], sh);
    return r;
}

enum { K_LOCAL = 0, K_REMOTE = 1, K_FULL = 2 };
struct Unit { int pi, pj; int need_x, need_w; int kind; int special; };

}

__global__ __launch_bounds__(512, 2) void fused_kernel(const Params P) {
    extern __shared__ __attribute__((aligned(16))) unsigned char shm_[];
    LAS unsigned char* lds = (LAS unsigned char*)shm_;
    const int tid = threadIdx.x;
    const int rank = P.rank;
    char* myws = P.ws[rank];
    unsigned* tmo = (unsigned*)(myws + WS_TMO);
    const int wid = __builtin_amdgcn_readfirstlane(tid >> 6), lane = tid & 63;
    const int bid = blockIdx.x;

    if (P.phase == 0) {
        LAS unsigned* sred = (LAS unsigned*)(lds + 2 * HTB);
        const bool isx = bid < PRE_WG / 2;
        f32x4 r[4][8];
#pragma unroll
        for (int i = 0; i < 4; ++i) {
            const int u = bid * 4 + i, blk = u >> 4, kk = u & 15;
            if (isx) {
#pragma unroll
                for (int c = 0; c < 4; ++c) {
                    int R, C; stage_rc(tid * 16 + (c & 1) * 8192, R, C);
                    const float* s = P.x + (size_t)(blk * 256 + (c >> 1) * 128 + R) * KS + kk * 64 + C;
                    r[i][2 * c] = __builtin_nontemporal_load((const f32x4*)s); r[i][2 * c + 1] = __builtin_nontemporal_load((const f32x4*)(s + 4));
                }
            } else {
                const float* s = P.w + (size_t)(kk * 64 + wid * 8) * NN + (blk - 32) * 256 + lane * 4;
#pragma unroll
                for (int e = 0; e < 8; ++e) r[i][e] = __builtin_nontemporal_load((const f32x4*)(s + (size_t)e * NN));
            }
        }
        unsigned m = 0;
#pragma unroll
        for (int i = 0; i < 4; ++i)
#pragma unroll
            for (int e = 0; e < 8; ++e)
#pragma unroll
                for (int j = 0; j < 4; ++j) m = max(m, __builtin_bit_cast(unsigned, r[i][e][j]) & 0x7fffffffu);
#pragma unroll
        for (int o = 32; o > 0; o >>= 1) m = max(m, (unsigned)__shfl_xor((int)m, o));
        if (lane == 0) sred[wid] = m;
        __syncthreads();
        if (tid == 0) {
#pragma unroll
            for (int i = 1; i < 8; ++i) m = max(m, sred[i]);
            __hip_atomic_store((unsigned*)(myws + WS_AMAXP) + bid, m, __ATOMIC_RELAXED, __HIP_MEMORY_SCOPE_AGENT);
            asm volatile("s_waitcnt vmcnt(0)" ::: "memory");
            sred[8] = __hip_atomic_fetch_add((unsigned*)(myws + WS_CTR0), 1u, __ATOMIC_RELEASE, __HIP_MEMORY_SCOPE_AGENT);
        }
        __syncthreads();
        const unsigned ticket = __builtin_amdgcn_readfirstlane(sred[8]);
        const unsigned call = ticket / PRE_WG + 1;
        if (ticket % PRE_WG == PRE_WG - 1) {
            __builtin_amdgcn_fence(__ATOMIC_ACQUIRE, "agent");
            asm volatile("s_waitcnt vmcnt(0)" ::: "memory");
            unsigned v = (tid < PRE_WG) ? ld_agent((const unsigned*)(myws + WS_AMAXP) + tid) : 0u;
#pragma unroll
            for (int o = 32; o > 0; o >>= 1) v = max(v, (unsigned)__shfl_xor((int)v, o));
            __syncthreads();
            if (lane == 0) sred[wid] = v;
            __syncthreads();
            if (tid < NR) {
                const unsigned ax = max(sred[0], sred[1]), aw = max(sred[2], sred[3]);
                unsigned long long* slot = (unsigned long long*)(P.ws[tid] + WS_XCHG + (size_t)rank * 128);
                __hip_atomic_store(slot, ((unsigned long long)call << 32) | ax, __ATOMIC_RELAXED, __HIP_MEMORY_SCOPE_SYSTEM);
                __hip_atomic_store(slot + 1, ((unsigned long long)call << 32) | aw, __ATOMIC_RELAXED, __HIP_MEMORY_SCOPE_SYSTEM);
            }
            __syncthreads();
        }
        if (wid == 0) {
            const unsigned long long* slot = (const unsigned long long*)(myws + WS_XCHG + (size_t)(lane >> 1 & 7) * 128) + (lane & 1);
            unsigned long long v = 0;
            const unsigned long long t0 = __builtin_amdgcn_s_memrealtime();
            for (;;) {
                v = __hip_atomic_load(slot, __ATOMIC_RELAXED, __HIP_MEMORY_SCOPE_SYSTEM);
                if (__all((unsigned)(v >> 32) >= call)) break;
                if (ld_agent(tmo) == call || __builtin_amdgcn_s_memrealtime() - t0 > DEADLINE) {
                    if (lane == 0) __hip_atomic_store(tmo, call, __ATOMIC_RELAXED, __HIP_MEMORY_SCOPE_AGENT);
                    break;
                }
                __builtin_amdgcn_s_sleep(1);
            }
            unsigned a = (unsigned)v;
            a = max(a, (unsigned)__shfl_xor((int)a, 2)); a = max(a, (unsigned)__shfl_xor((int)a, 4)); a = max(a, (unsigned)__shfl_xor((int)a, 8));
            const unsigned ax = __shfl((int)a, 0), aw = __shfl((int)a, 1);
            if (lane == 0) {
                sred[9] = (unsigned)(14 - scale_exp(ax)); sred[10] = (unsigned)(14 - scale_exp(aw));
                if (bid == 0) { ((int*)(myws + WS_SCL))[0] = 14 - scale_exp(ax); ((int*)(myws + WS_SCL))[1] = 14 - scale_exp(aw); }
            }
        }
        __syncthreads();
        const int shx = (int)__builtin_amdgcn_readfirstlane(sred[9]), shw = (int)__builtin_amdgcn_readfirstlane(sred[10]);
        char* img = myws + WS_STAGE + (size_t)(call & 1) * STAGE_PAR + (size_t)rank * 64 * BLKB + (size_t)bid * 4 * 32768 + tid * 16;
#pragma unroll
        for (int i = 0; i < 4; ++i) {
            char* dst = img + i * 32768;
            if (isx) {
#pragma unroll
                for (int c = 0; c < 4; ++c) *(h8*)(dst + c * 8192) = cvt8(r[i][2 * c], r[i][2 * c + 1], shx);
            } else {
                __syncthreads();
#pragma unroll
                for (int c = 0; c < 4; ++c) {
                    const f32x4 a = {r[i][0][c], r[i][1][c], r[i][2][c], r[i][3][c]}, b2 = {r[i][4][c], r[i][5][c], r[i][6][c], r[i][7][c]};
                    const int nn = lane * 4 + c;
                    *(LAS h8*)(lds + (nn >> 7) * HTB + lds_byte(nn & 127, wid * 8)) = cvt8(a, b2, shw);
                }
                __syncthreads();
#pragma unroll
                for (int c = 0; c < 4; ++c) *(h8*)(dst + c * 8192) = *(const LAS h8*)(lds + tid * 16 + c * 8192);
            }
        }
        return;
    }

    LAS unsigned* sctl = (LAS unsigned*)(lds + 8 * HTB);
#define STAMP() do {} while (0)
    STAMP();
    if (tid == 0) sctl[0] = __hip_atomic_fetch_add((unsigned*)(myws + WS_CTR2), 1u, __ATOMIC_RELAXED, __HIP_MEMORY_SCOPE_AGENT) / NWG + 1;
    __syncthreads();
    const unsigned call = __builtin_amdgcn_readfirstlane(sctl[0]);
    const int osh = -(__builtin_amdgcn_readfirstlane(((const int*)(myws + WS_SCL))[0]) + __builtin_amdgcn_readfirstlane(((const int*)(myws + WS_SCL))[1]));
    const size_t stage_off = WS_STAGE + (size_t)(call & 1) * STAGE_PAR;

    const int xcd = bid & 7, slot = bid >> 3;
    const bool is_comm = bid < NCOMM;
    if (is_comm) {
        for (int n = bid; n < NITEM; n += NCOMM) {
            int blk, q; item_of(n, blk, q);
            const unsigned dm = dest_mask(blk) & ~(1u << rank);
            if (dm == 0u) continue;
            const size_t ioff = stage_off + ((size_t)rank * 64 + blk) * BLKB + (size_t)(4 * q) * 32768 + tid * 16;
            u32x4 v0 = __builtin_nontemporal_load((const u32x4*)(myws + ioff)), v1 = __builtin_nontemporal_load((const u32x4*)(myws + ioff + 8192));
#pragma unroll 1
            for (int c = 0; c < 8; ++c) {
                const u32x4 w0 = v0, w1 = v1;
                if (c < 7) { v0 = __builtin_nontemporal_load((const u32x4*)(myws + ioff + (c + 1) * 16384)); v1 = __builtin_nontemporal_load((const u32x4*)(myws + ioff + (c + 1) * 16384 + 8192)); }
#pragma unroll
                for (int d = 1; d < NR; ++d) {
                    const int p = (rank + d) & 7;
                    if ((dm >> p) & 1u) {
                        char* dst = P.ws[p] + ioff + c * 16384;
                        st_sys(dst, w0); st_sys(dst + 8192, w1);
                    }
                }
            }
            asm volatile("s_waitcnt vmcnt(0)" ::: "memory");
            __syncthreads();
            if (wid == 0 && lane < NR && ((dm >> lane) & 1u))
                __hip_atomic_store((unsigned*)(P.ws[lane] + WS_FLAGS + (size_t)item_index(blk, q) * 128) + rank, call, __ATOMIC_RELAXED, __HIP_MEMORY_SCOPE_SYSTEM);
            STAMP();
        }
        for (int z = bid; z < 4 * NSPEC; z += NCOMM) {
            const int pos = z >> 2, sq = z & 3;
            if (wid == 0) {
                const unsigned* f = (const unsigned*)(myws + WS_SDONE + (size_t)pos * 128);
                const unsigned long long t0 = __builtin_amdgcn_s_memrealtime();
                while (ld_agent(f) < call * 8u) {
                    if (ld_agent(tmo) == call || __builtin_amdgcn_s_memrealtime() - t0 > DEADLINE) {
                        if (lane == 0) __hip_atomic_store(tmo, call, __ATOMIC_RELAXED, __HIP_MEMORY_SCOPE_AGENT);
                        break;
                    }
                    __builtin_amdgcn_s_sleep(2);
                }
                __builtin_amdgcn_fence(__ATOMIC_ACQUIRE, "agent");
                asm volatile("s_waitcnt vmcnt(0)" ::: "memory");
            }
            __syncthreads();
            const int st = NSPEC * rank + pos;
            const size_t ooff = ((size_t)((SROW + (st >> 5)) * 256 + sq * 64 + wid * 8) * NN + (st & 31) * 256 + lane * 4) * 4;
            const bool bad = (ld_agent(tmo) == call);
#pragma unroll 1
            for (int r = 0; r < 8; r += 2) {
                u32x4 w0 = *(const u32x4*)((const char*)P.out[rank] + ooff + (size_t)r * NN * 4), w1 = *(const u32x4*)((const char*)P.out[rank] + ooff + (size_t)(r + 1) * NN * 4);
                if (bad) { w0 = (u32x4){0x7fc00000u, 0x7fc00000u, 0x7fc00000u, 0x7fc00000u}; w1 = w0; }
#pragma unroll
                for (int d = 1; d < NR; ++d) {
                    char* dst = (char*)P.out[(rank + d) & 7] + ooff + (size_t)r * NN * 4;
                    st_sys(dst, w0); st_sys(dst + (size_t)NN * 4, w1);
                }
            }
            asm volatile("s_waitcnt vmcnt(0)" ::: "memory");
            __syncthreads();
            if (wid == 0 && lane < NR && lane != rank)
                __hip_atomic_fetch_add((unsigned*)(P.ws[lane] + WS_SPEC + (size_t)rank * 128), 1u, __ATOMIC_RELAXED, __HIP_MEMORY_SCOPE_SYSTEM);
            STAMP();
        }
    }

    const int pos = xcd * 30 + (slot - NCOMM / 8);
    auto tile_of = [&](int r, int& pi, int& pj, int& special) -> bool {
        special = 0;
        if (r == 0) { pi = (pos % 60) >> 2; pj = (pos / 60) * 4 + (pos & 3); return true; }
        int b;
        if (r == 1) {
            if (pos < NSPEC) { const int st = NSPEC * rank + pos; pi = SROW + (st >> 5); pj = st & 31; special = 1; return true; }
            b = pos - NSPEC;
        } else {
            if (pos >= NSPEC) { const int c = pos - NSPEC; pi = 15 + c % 6; pj = c / 6; return c < 192; }
            b = 240 - NSPEC + pos;
        }
        pi = (b % 60) >> 2; pj = 16 + (b / 60) * 4 + (b & 3);
        return true;
    };
    int pi0, pj0, pi1, pj1, pi2, pj2, sp0, sp1, sp2;
    tile_of(0, pi0, pj0, sp0); tile_of(1, pi1, pj1, sp1);
    const bool has2 = tile_of(2, pi2, pj2, sp2);
    const int nunits = is_comm ? 0 : (has2 ? 5 : 3);
    auto get_unit = [&](int ui) -> Unit {
        Unit u;
        if (!has2 && ui > 0) ui += 1;
        u.special = 0; u.need_x = 1; u.need_w = 1;
        if (ui == 0) { u.pi = pi1; u.pj = pj1; u.kind = K_LOCAL; }
        else if (ui == 1) { u.pi = pi2; u.pj = pj2; u.kind = K_LOCAL; }
        else if (ui == 2) { u.pi = pi0; u.pj = pj0; u.kind = K_FULL; }
        else if (ui == 3) { u.pi = pi1; u.pj = pj1; u.kind = K_REMOTE; u.special = sp1; u.need_x = (pi1 != pi0); u.need_w = (pj1 != pj0); }
        else { u.pi = pi2; u.pj = pj2; u.kind = K_REMOTE; u.need_x = (pi2 != pi0) && (pi2 != pi1); u.need_w = (pj2 != pj0) && (pj2 != pj1); }
        return u;
    };
    auto koff = [&](int kind, int kt) -> size_t {
        if (kind == K_FULL) { if (kt < 16) kind = K_LOCAL; else { kt -= 16; kind = K_REMOTE; } }
        if (kind == K_LOCAL) return ((size_t)(rank * 64) * 16 + kt) * 32768ull;
        const int q = kt / 28, r = kt - q * 28, p = (rank + 1 + (r >> 2)) & 7;
        return ((size_t)(p * 64) * 16 + 4 * q + (r & 3)) * 32768ull;
    };

    if (nunits > 0) {
        const int wr = wid >> 2, wc = wid & 3, fr = lane & 15, fq = lane >> 4;
        const unsigned voff0 = tid * 16, voff1 = tid * 16 + 8192;
        const unsigned ldsw = (unsigned)wid * 1024u;
        const int aoff = lds_byte(wr * 64 + fr, fq * 8), boff = lds_byte(wc * 32 + fr, fq * 8);
        const char* stage = myws + stage_off;
#define SA(b, h) (((b) * 2 + (h)) * HTB)
#define SB(b, h) ((4 + (b) * 2 + (h)) * HTB)
#define STAGE(bufoff, gbase) do { \
        __builtin_amdgcn_global_load_lds((const unsigned*)((const char*)(gbase) + voff0), (LAS unsigned*)(lds + (bufoff) + ldsw), 16, 0, 0); \
        __builtin_amdgcn_global_load_lds((const unsigned*)((const char*)(gbase) + voff1), (LAS unsigned*)(lds + (bufoff) + ldsw + 8192), 16, 0, 0); } while (0)
#define LDA(dst, b, h) do { _Pragma("unroll") for (int m = 0; m < 4; ++m) _Pragma("unroll") for (int k = 0; k < 2; ++k) dst[m][k] = *(const LAS h8*)(lds + SA(b, h) + aoff + m * 2048 + k * 1024); } while (0)
#define LDB(dst, b, h) do { _Pragma("unroll") for (int n = 0; n < 2; ++n) _Pragma("unroll") for (int k = 0; k < 2; ++k) dst[n][k] = *(const LAS h8*)(lds + SB(b, h) + boff + n * 2048 + k * 1024); } while (0)
#define MMA(ai, bj, At, Bt) do { __builtin_amdgcn_s_setprio(1); _Pragma("unroll") for (int m = 0; m < 4; ++m) _Pragma("unroll") for (int n = 0; n < 2; ++n) _Pragma("unroll") for (int k = 0; k < 2; ++k) \
        acc[ai][bj][m][n] = __builtin_amdgcn_mfma_f32_16x16x32_f16(Bt[n][k], At[m][k], acc[ai][bj][m][n], 0, 0, 0); __builtin_amdgcn_s_setprio(0); } while (0)
#define WAIT_V(n) asm volatile("s_waitcnt vmcnt(" #n ")" ::: "memory")
#define WAIT_L(n) asm volatile("s_waitcnt lgkmcnt(" #n ")" ::: "memory")
#define BAR __builtin_amdgcn_s_barrier()
#define SCHED __builtin_amdgcn_sched_barrier(0)
        auto ready = [&](const Unit& u, int q) {
            if (!(u.need_x | u.need_w)) return;
            if (wid == 0) {
                const int it = item_index((lane < 8) ? u.pi : 32 + u.pj, q);
                const bool need = (lane < 16) && ((lane & 7) != rank) && ((lane < 8) ? u.need_x : u.need_w);
                const unsigned* f = (const unsigned*)(myws + WS_FLAGS + (size_t)it * 128) + (lane & 7);
                const unsigned long long t0 = __builtin_amdgcn_s_memrealtime();
                for (;;) {
                    const unsigned v = need ? ld_sys(f) : call;
                    if (__all(v >= call)) break;
                    if (ld_agent(tmo) == call || __builtin_amdgcn_s_memrealtime() - t0 > DEADLINE) {
                        if (lane == 0) __hip_atomic_store(tmo, call, __ATOMIC_RELAXED, __HIP_MEMORY_SCOPE_AGENT);
                        break;
                    }
                    __builtin_amdgcn_s_sleep(1);
                }
                __builtin_amdgcn_fence(__ATOMIC_ACQUIRE, "");
                asm volatile("s_waitcnt vmcnt(0)" ::: "memory");
            }
            asm volatile("" ::: "memory"); BAR; asm volatile("" ::: "memory");
            STAMP();
        };

        f32x4 acc[2][2][4][2];
#pragma unroll
        for (int a = 0; a < 2; ++a)
#pragma unroll
            for (int b = 0; b < 2; ++b)
#pragma unroll
                for (int m = 0; m < 4; ++m)
#pragma unroll
                    for (int n = 0; n < 2; ++n) acc[a][b][m][n] = (f32x4){0.f, 0.f, 0.f, 0.f};
        h8 At[4][2], B0[2][2], B1[2][2];
        Unit cur = get_unit(0);
        const char* cA = stage + (size_t)cur.pi * BLKB; const char* cB = stage + (size_t)(32 + cur.pj) * BLKB;
        {
            const size_t o0 = koff(cur.kind, 0);
            STAGE(SB(0, 0), cB + o0); STAGE(SA(0, 0), cA + o0); STAGE(SB(0, 1), cB + o0 + HTB); STAGE(SA(0, 1), cA + o0 + HTB);
            if (wr == 1) BAR;
            WAIT_V(4); BAR;
            STAGE(SB(1, 0), cB + o0 + 32768); STAGE(SA(1, 0), cA + o0 + 32768); STAGE(SB(1, 1), cB + o0 + 32768 + HTB);
            WAIT_V(6); BAR;
        }
        for (int ui = 0;; ++ui) {
            const bool has_next = ui + 1 < nunits;
            const Unit nxt = has_next ? get_unit(ui + 1) : cur;
            const char* nA = stage + (size_t)nxt.pi * BLKB; const char* nB = stage + (size_t)(32 + nxt.pj) * BLKB;
            const int nt = (cur.kind == K_LOCAL) ? 16 : (cur.kind == K_REMOTE) ? 112 : 128;
            const int rbase = (cur.kind == K_FULL) ? 16 : 0;
            for (int t = 0; t < nt; t += 2) {
                const bool last = (t == nt - 2);
                const char* a1 = cA + koff(cur.kind, t) + 32768;
                const size_t o2 = last ? koff(nxt.kind, 0) : koff(cur.kind, t + 2);
                const char* a2 = (last ? nA : cA) + o2; const char* b2 = (last ? nB : cB) + o2;
                const char* a3 = a2 + 32768; const char* b3 = b2 + 32768;
                if (last) { if (has_next && nxt.kind == K_REMOTE) ready(nxt, 0); }
                else if (cur.kind != K_LOCAL && t + 2 >= rbase && (t + 2 - rbase) % 28 == 0) ready(cur, (t + 2 - rbase) / 28);
                LDB(B0, 0, 0); SCHED; LDA(At, 0, 0); STAGE(SA(1, 1), a1 + HTB);
                WAIT_L(8); BAR; WAIT_L(0); MMA(0, 0, At, B0); BAR; SCHED;
                LDB(B1, 0, 1); STAGE(SB(0, 0), b2);
                BAR; WAIT_L(0); MMA(0, 1, At, B1); BAR;
                LDA(At, 0, 1); STAGE(SA(0, 0), a2);
                BAR; WAIT_L(0); MMA(1, 0, At, B0); BAR; SCHED;
                STAGE(SB(0, 1), b2 + HTB);
                WAIT_V(6); BAR; MMA(1, 1, At, B1); BAR;
                LDB(B0, 1, 0); SCHED; LDA(At, 1, 0); STAGE(SA(0, 1), a2 + HTB);
                WAIT_L(8); BAR; WAIT_L(0); MMA(0, 0, At, B0); BAR; SCHED;
                LDB(B1, 1, 1); STAGE(SB(1, 0), b3);
                BAR; WAIT_L(0); MMA(0, 1, At, B1); BAR;
                LDA(At, 1, 1); STAGE(SA(1, 0), a3);
                BAR; WAIT_L(0); MMA(1, 0, At, B0); BAR; SCHED;
                STAGE(SB(1, 1), b3 + HTB);
                WAIT_V(6); BAR; MMA(1, 1, At, B1); BAR;
            }
            {
                const size_t off0 = (size_t)(cur.pi * 256 + wr * 64 + fr) * NN + cur.pj * 256 + wc * 32 + 4 * fq;
                const bool bad = (cur.kind != K_LOCAL) && (ld_agent(tmo) == call);
                const float qnan = __builtin_nanf("");
                const int sh = (cur.kind == K_LOCAL) ? 0 : osh;
                if (cur.kind == K_REMOTE) {
                    const float* C = P.out[rank] + off0;
#pragma unroll
                    for (int ai = 0; ai < 2; ++ai)
#pragma unroll
                        for (int m = 0; m < 4; ++m) { const float* rowp = C + (size_t)(ai * 128 + m * 16) * NN;
#pragma unroll
                            for (int bj = 0; bj < 2; ++bj)
#pragma unroll
                                for (int n = 0; n < 2; ++n) acc[ai][bj][m][n] += *(const f32x4*)(rowp + bj * 128 + n * 16);
                            asm volatile("" : "+v"(acc[ai][0][m][0]), "+v"(acc[ai][0][m][1]), "+v"(acc[ai][1][m][0]), "+v"(acc[ai][1][m][1]) :: "memory"); }
                }
#pragma unroll
                for (int ai = 0; ai < 2; ++ai)
#pragma unroll
                    for (int bj = 0; bj < 2; ++bj)
#pragma unroll
                        for (int m = 0; m < 4; ++m)
#pragma unroll
                            for (int n = 0; n < 2; ++n)
#pragma unroll
                                for (int j = 0; j < 4; ++j) acc[ai][bj][m][n][j] = bad ? qnan : __builtin_amdgcn_ldexpf(acc[ai][bj][m][n][j], sh);
                float* C = P.out[rank] + off0;
                if (cur.special) {
#pragma unroll
                    for (int ai = 0; ai < 2; ++ai)
#pragma unroll
                        for (int m = 0; m < 4; ++m) { float* rowp = C + (size_t)(ai * 128 + m * 16) * NN;
#pragma unroll
                            for (int bj = 0; bj < 2; ++bj)
#pragma unroll
                                for (int n = 0; n < 2; ++n) st_sys((char*)(rowp + bj * 128 + n * 16), __builtin_bit_cast(u32x4, acc[ai][bj][m][n])); }
                    asm volatile("s_waitcnt vmcnt(0)" ::: "memory");
                    if (lane == 0) __hip_atomic_fetch_add((unsigned*)(myws + WS_SDONE + (size_t)pos * 128), 1u, __ATOMIC_RELAXED, __HIP_MEMORY_SCOPE_AGENT);
                } else {
#pragma unroll
                    for (int ai = 0; ai < 2; ++ai)
#pragma unroll
                        for (int m = 0; m < 4; ++m) { float* rowp = C + (size_t)(ai * 128 + m * 16) * NN;
#pragma unroll
                            for (int bj = 0; bj < 2; ++bj)
#pragma unroll
                                for (int n = 0; n < 2; ++n) __builtin_nontemporal_store(acc[ai][bj][m][n], (f32x4*)(rowp + bj * 128 + n * 16)); }
                }
            }
            STAMP();
            if (!has_next) break;
#pragma unroll
            for (int a = 0; a < 2; ++a)
#pragma unroll
                for (int b = 0; b < 2; ++b)
#pragma unroll
                    for (int m = 0; m < 4; ++m)
#pragma unroll
                        for (int n = 0; n < 2; ++n) acc[a][b][m][n] = (f32x4){0.f, 0.f, 0.f, 0.f};
            cur = nxt; cA = nA; cB = nB;
        }
        WAIT_V(0);
        if (wr == 0) BAR;
        BAR;
    }

    if (bid == NCOMM - 1 && wid == 0) {
        const unsigned need = call * (unsigned)(4 * NSPEC);
        const unsigned* f = (const unsigned*)(myws + WS_SPEC + (size_t)(lane & 7) * 128);
        const bool mine = (lane < NR) && (lane != rank);
        const unsigned long long t0 = __builtin_amdgcn_s_memrealtime();
        bool to = false;
        for (;;) {
            const unsigned v = mine ? ld_sys(f) : need;
            if (__all(v >= need)) break;
            if (ld_agent(tmo) == call || __builtin_amdgcn_s_memrealtime() - t0 > DEADLINE) {
                if (lane == 0) __hip_atomic_store(tmo, call, __ATOMIC_RELAXED, __HIP_MEMORY_SCOPE_AGENT);
                to = true;
                break;
            }
            __builtin_amdgcn_s_sleep(4);
        }
        __builtin_amdgcn_fence(__ATOMIC_ACQUIRE, "");
        if (to && lane == 0) P.out[rank][0] = __builtin_nanf("");
    }
    STAMP();
}

extern "C" void kernel_launch(int rank, int nranks, ncclComm_t comm,
                              hipStream_t stream,
                              void* const* d_in, const int* in_sizes, int n_in,
                              void* d_out, int out_size,
                              const int* in_dtypes, int out_dtype,
                              void* const* d_out_all,
                              void* const* d_ws_all, size_t ws_bytes) {
    if (nranks != NR || ws_bytes < WS_NEED || (unsigned)rank >= (unsigned)NR) { fprintf(stderr, "unsupported configuration\n"); return; }
    static bool init = false;
    if (!init) {
        HIP_CHECK(hipFuncSetAttribute((const void*)fused_kernel, hipFuncAttributeMaxDynamicSharedMemorySize, 8 * HTB + 256));
        init = true;
    }
    Params P;
    P.x = (const float*)d_in[0];
    P.w = (const float*)d_in[1];
    for (int p = 0; p < NR; ++p) { P.out[p] = (float*)d_out_all[p]; P.ws[p] = (char*)d_ws_all[p]; }
    P.rank = rank;
    P.phase = 0;
    fused_kernel<<<PRE_WG, 512, 2 * HTB + 256, stream>>>(P);
    P.phase = 1;
    fused_kernel<<<NWG, 512, 8 * HTB + 256, stream>>>(P);
    HIP_CHECK(hipGetLastError());
}
```
